```python
import numpy as np

kernel_name = "gpu_a2a_gemm_m2048_k2048_n2048_bf16_relu"


ABSMAX_RTOL = 0.008

COMPUTE_DTYPE = "bfloat16"
EPILOGUE = "relu"
IN_QUANT = None
OUT_QUANT = None
IN_DTYPES = {"x": "bfloat16", "w_mat": "bfloat16"}
WORLD_SIZE = 8

def _to_bf16(x):
    u = np.ascontiguousarray(x, dtype=np.float32).view(np.uint32)
    return ((u + 0x7FFF + ((u >> 16) & 1)) & 0xFFFF0000).view(np.float32)

def setup_inputs():

    x = _to_bf16(np.random.randn(2048, 2048).astype(np.float32))
    w_mat = _to_bf16((np.random.randn(2048, 2048) / np.sqrt(2048)).astype(np.float32))
    return {"x": x, "w_mat": w_mat}

def _epilogue(y):
    return np.maximum(y, 0.0).astype(np.float32)

def reference(x, w_mat):
    y = (x @ w_mat).astype(np.float32)
    return _epilogue(y)
```

```cpp
#include <hip/hip_runtime.h>
#include <rccl/rccl.h>
#include <cstdint>
#include <cstdio>

typedef __attribute__((ext_vector_type(8))) __bf16 bf16x8;
typedef __attribute__((ext_vector_type(4))) float f32x4;
typedef __attribute__((ext_vector_type(4))) short s16x4;
typedef __attribute__((ext_vector_type(4))) unsigned u32x4;

constexpr int NR = 8;
constexpr size_t STAGE_BYTES = 1u << 20;
constexpr size_t FLAG_OFF = 2 * STAGE_BYTES;
constexpr size_t CTR_OFF = FLAG_OFF + 65536;
constexpr size_t TMO_OFF = CTR_OFF + 65536;
constexpr size_t WS_NEED = TMO_OFF + 4096;
constexpr unsigned long long DEADLINE = 30000000ull;
constexpr int LDS_BYTES = 2048 * 64;

struct Params {
  const uint16_t* x; const uint16_t* w; uint16_t* out;
  char* ws[NR]; int rank;
};

__device__ __forceinline__ s16x4 trread(const char* p) {
  return __builtin_bit_cast(s16x4, __builtin_amdgcn_ds_read_tr16_b64_v4i16(
      (__attribute__((address_space(3))) s16x4*)(p)));
}

__device__ __forceinline__ uint16_t f2bf(float f) {
  unsigned u = __float_as_uint(f);
  u += 0x7FFFu + ((u >> 16) & 1u);
  return (uint16_t)(u >> 16);
}

__global__ __launch_bounds__(256, 1) void a2a_gemm_kernel(Params p) {
  extern __shared__ __attribute__((aligned(16))) char smem[];
  const int b = blockIdx.x, tid = threadIdx.x, lane = tid & 63, wv = tid >> 6;
  const int rank = p.rank;
  char* myws = p.ws[rank];
  unsigned* cctr = (unsigned*)(myws + CTR_OFF + (size_t)b * 128);
  const unsigned call = __hip_atomic_load(cctr, __ATOMIC_RELAXED, __HIP_MEMORY_SCOPE_AGENT) + 1u;
  const size_t par = (size_t)(call & 1u) * STAGE_BYTES;
  unsigned* tmo = (unsigned*)(myws + TMO_OFF);

  if (b < 224) {
    const int s = b / 56, rem = b % 56, pi = rem % 7, part = rem / 7;
    const unsigned peer = (unsigned)(rank + 1 + pi) & 7u;
    const int rg = part * 2 + (wv >> 1), ks = s * 2 + (wv & 1);
    const u32x4 v = *(const u32x4*)(p.x + (size_t)(peer * 256 + rg * 16 + (lane & 15)) * 256 + ks * 32 + (lane >> 4) * 8);
    char* pw = p.ws[peer];
    *(u32x4*)(pw + par + (size_t)((rank * 16 + rg) * 8 + ks) * 1024 + lane * 16) = v;
    asm volatile("s_waitcnt vmcnt(0)" ::: "memory");
    __syncthreads();
    if (tid == 0) {
      __builtin_amdgcn_fence(__ATOMIC_RELEASE, "");
      asm volatile("s_waitcnt vmcnt(0)" ::: "memory");
      __hip_atomic_fetch_add((unsigned*)(pw + FLAG_OFF + (size_t)(rank * 4 + s) * 128), 1u,
                             __ATOMIC_RELAXED, __HIP_MEMORY_SCOPE_SYSTEM);
    }
  }

  const int xcd = b & 7, idx = b >> 3;
  const int nt = xcd * 8 + (idx & 7), mt = idx >> 3;
  {
    const int row = lane >> 2, piece = (lane & 3) ^ (((row >> 3) & 1) << 1);
    const uint16_t* wsrc = p.w + (size_t)row * 2048 + nt * 32 + piece * 8;
#pragma unroll 8
    for (int i = 0; i < 32; ++i) {
      const int blk = i * 4 + wv;
      __builtin_amdgcn_global_load_lds((const unsigned*)(wsrc + (size_t)blk * 16 * 2048),
          (__attribute__((address_space(3))) unsigned*)(smem + blk * 1024), 16, 0, 0);
    }
  }
  asm volatile("s_waitcnt vmcnt(0)" ::: "memory");
  __syncthreads();

  f32x4 acc0 = {0, 0, 0, 0}, acc1 = {0, 0, 0, 0};
  const int g = lane >> 4;
  const int boff0 = (8 * g + ((lane >> 2) & 3)) * 64 + ((g & 1) * 32) + (lane & 3) * 8;
  const int boff1 = boff0 ^ 32;
  auto step = [&](const u32x4& a, int kk) {
    const char* bp = smem + kk * 64;
    s16x4 l0 = trread(bp + boff0), h0 = trread(bp + boff0 + 256);
    s16x4 l1 = trread(bp + boff1), h1 = trread(bp + boff1 + 256);
    bf16x8 b0 = __builtin_bit_cast(bf16x8, __builtin_shufflevector(l0, h0, 0, 1, 2, 3, 4, 5, 6, 7));
    bf16x8 b1 = __builtin_bit_cast(bf16x8, __builtin_shufflevector(l1, h1, 0, 1, 2, 3, 4, 5, 6, 7));
    bf16x8 af = __builtin_bit_cast(bf16x8, a);
    acc0 = __builtin_amdgcn_mfma_f32_16x16x32_bf16(af, b0, acc0, 0, 0, 0);
    acc1 = __builtin_amdgcn_mfma_f32_16x16x32_bf16(af, b1, acc1, 0, 0, 0);
  };
  const int rg = mt * 4 + wv;
  {
    const uint16_t* xa = p.x + (size_t)(rank * 256 + rg * 16 + (lane & 15)) * 256 + (lane >> 4) * 8;
    u32x4 a[8];
#pragma unroll
    for (int k = 0; k < 8; ++k) a[k] = *(const u32x4*)(xa + k * 32);
#pragma unroll
    for (int k = 0; k < 8; ++k) step(a[k], rank * 256 + k * 32);
  }
  const unsigned target = 8u * call;
  const char* abase = myws + par + (size_t)rg * 8192 + lane * 16;
  bool dead = false;
  for (int s = 0; s < 4; ++s) {
    {
      const bool need = lane < 8 && lane != rank;
      unsigned* fl = (unsigned*)(myws + FLAG_OFF + (size_t)((lane & 7) * 4 + s) * 128);
      const unsigned long long t0 = __builtin_amdgcn_s_memrealtime();
      while (!dead) {
        unsigned v = need ? __hip_atomic_load(fl, __ATOMIC_RELAXED, __HIP_MEMORY_SCOPE_SYSTEM) : target;
        if (__all((int)(v - target) >= 0)) break;
        if (__hip_atomic_load(tmo, __ATOMIC_RELAXED, __HIP_MEMORY_SCOPE_AGENT) == call ||
            __builtin_amdgcn_s_memrealtime() - t0 > DEADLINE) {
          __hip_atomic_store(tmo, call, __ATOMIC_RELAXED, __HIP_MEMORY_SCOPE_AGENT);
          dead = true;
        }
        __builtin_amdgcn_s_sleep(1);
      }
      __builtin_amdgcn_fence(__ATOMIC_ACQUIRE, "");
    }
    if (dead) break;
    u32x4 a[14];
#pragma unroll
    for (int i = 0; i < 7; ++i) {
      const int src = (rank + 1 + i) & 7;
      a[2 * i] = *(const u32x4*)(abase + (size_t)src * 131072 + (s * 2) * 1024);
      a[2 * i + 1] = *(const u32x4*)(abase + (size_t)src * 131072 + (s * 2 + 1) * 1024);
    }
#pragma unroll
    for (int i = 0; i < 7; ++i) {
      const int src = (rank + 1 + i) & 7;
      step(a[2 * i], src * 256 + s * 64);
      step(a[2 * i + 1], src * 256 + s * 64 + 32);
    }
  }

  if (!dead) dead = __hip_atomic_load(tmo, __ATOMIC_RELAXED, __HIP_MEMORY_SCOPE_AGENT) == call;
  uint16_t* o = p.out + (size_t)(rg * 16 + g * 4) * 2048 + nt * 32 + (lane & 15);
#pragma unroll
  for (int r = 0; r < 4; ++r) {
    o[r * 2048] = dead ? (uint16_t)0x7FC0 : f2bf(fmaxf(acc0[r], 0.f));
    o[r * 2048 + 16] = dead ? (uint16_t)0x7FC0 : f2bf(fmaxf(acc1[r], 0.f));
  }
  if (tid == 0) __hip_atomic_store(cctr, call, __ATOMIC_RELAXED, __HIP_MEMORY_SCOPE_AGENT);
}

extern "C" void kernel_launch(int rank, int nranks, ncclComm_t comm, hipStream_t stream,
                              void* const* d_in, const int* in_sizes, int n_in,
                              void* d_out, int out_size, const int* in_dtypes, int out_dtype,
                              void* const* d_out_all, void* const* d_ws_all, size_t ws_bytes) {
  if (nranks != NR || ws_bytes < WS_NEED) return;
  Params p;
  p.x = (const uint16_t*)d_in[0]; p.w = (const uint16_t*)d_in[1]; p.out = (uint16_t*)d_out;
  for (int i = 0; i < NR; ++i) p.ws[i] = (char*)d_ws_all[i];
  p.rank = rank;
  hipLaunchKernelGGL(a2a_gemm_kernel, dim3(256), dim3(256), LDS_BYTES, stream, p);
}
```

```cpp
#include <hip/hip_runtime.h>
#include <rccl/rccl.h>
#include <cstdint>
#include <cstdio>

typedef __attribute__((ext_vector_type(8))) __bf16 bf16x8;
typedef __attribute__((ext_vector_type(4))) float f32x4;
typedef __attribute__((ext_vector_type(4))) short s16x4;
typedef __attribute__((ext_vector_type(4))) unsigned u32x4;

#define NOACQ
#define NOREL
#define LDM "sc0 sc1"
constexpr int NR = 8;
constexpr size_t STAGE_BYTES = 1u << 20;
constexpr size_t FLAG_OFF = 2 * STAGE_BYTES;
constexpr size_t CTR_OFF = FLAG_OFF + 65536;
constexpr size_t TMO_OFF = CTR_OFF + 65536;
constexpr size_t WS_NEED = TMO_OFF + 4096;
constexpr unsigned long long DEADLINE = 30000000ull;
constexpr int LDS_BYTES = 2048 * 64;

struct Params {
  const uint16_t* x; const uint16_t* w; uint16_t* out;
  char* ws[NR]; int rank;
};

__device__ __forceinline__ s16x4 trread(const char* p) {
  return __builtin_bit_cast(s16x4, __builtin_amdgcn_ds_read_tr16_b64_v4i16(
      (__attribute__((address_space(3))) s16x4*)(p)));
}

__device__ __forceinline__ uint16_t f2bf(float f) {
  unsigned u = __float_as_uint(f);
  u += 0x7FFFu + ((u >> 16) & 1u);
  return (uint16_t)(u >> 16);
}

#define STAMP(i)
__global__ __launch_bounds__(256, 1) void a2a_gemm_kernel(Params p) {
  extern __shared__ __attribute__((aligned(16))) char smem[];
  const int b = blockIdx.x, tid = threadIdx.x, lane = tid & 63, wv = tid >> 6;
  const int rank = p.rank;
  STAMP(0);
  char* myws = p.ws[rank];
  unsigned* cctr = (unsigned*)(myws + CTR_OFF + (size_t)b * 128);
  const unsigned call = __hip_atomic_load(cctr, __ATOMIC_RELAXED, __HIP_MEMORY_SCOPE_AGENT) + 1u;
  const size_t par = (size_t)(call & 1u) * STAGE_BYTES;
  unsigned* tmo = (unsigned*)(myws + TMO_OFF);

  const int xcd = b & 7, idx = b >> 3;
  const int nt = xcd * 8 + (idx & 7), mt = idx >> 3;
  auto WLOAD = [&]() {
    const int row = lane >> 2, piece = (lane & 3) ^ (((row >> 3) & 1) << 1);
    const uint16_t* wsrc = p.w + (size_t)row * 2048 + nt * 32 + piece * 8;
#pragma unroll 8
    for (int i = 0; i < 32; ++i) {
      const int blk = i * 4 + wv;
      __builtin_amdgcn_global_load_lds((const unsigned*)(wsrc + (size_t)blk * 16 * 2048),
          (__attribute__((address_space(3))) unsigned*)(smem + blk * 1024), 16, 0, 0);
    }
  };
  if (b < 224) {
    const int s = b / 56, rem = b % 56, pi = rem % 7, part = rem / 7;
    const unsigned peer = (unsigned)(rank + 1 + pi) & 7u;
    const int rg = part * 2 + (wv >> 1), ks = s * 2 + (wv & 1);
    const u32x4 v = *(const u32x4*)(p.x + (size_t)(peer * 256 + rg * 16 + (lane & 15)) * 256 + ks * 32 + (lane >> 4) * 8);
    char* pw = p.ws[peer];
    asm volatile("global_store_dwordx4 %0, %1, off sc0 sc1" :: "v"(pw + par + (size_t)((rank * 16 + rg) * 8 + ks) * 1024 + lane * 16), "v"(v) : "memory");
    WLOAD();
    asm volatile("s_waitcnt vmcnt(32)" ::: "memory");
    __builtin_amdgcn_s_barrier();
    if (tid == 0) {
      __hip_atomic_fetch_add((unsigned*)(pw + FLAG_OFF + (size_t)(rank * 4 + s) * 128), 1u,
                             __ATOMIC_RELAXED, __HIP_MEMORY_SCOPE_SYSTEM);
    }
  } else WLOAD();

  STAMP(1);
  asm volatile("s_waitcnt vmcnt(0)" ::: "memory");
  __syncthreads();
  STAMP(2);
  f32x4 acc0 = {0, 0, 0, 0}, acc1 = {0, 0, 0, 0};
  const int g = lane >> 4;
  const int boff0 = (8 * g + ((lane >> 2) & 3)) * 64 + ((g & 1) * 32) + (lane & 3) * 8;
  const int boff1 = boff0 ^ 32;
  auto step = [&](const u32x4& a, int kk) {
    const char* bp = smem + kk * 64;
    s16x4 l0 = trread(bp + boff0), h0 = trread(bp + boff0 + 256);
    s16x4 l1 = trread(bp + boff1), h1 = trread(bp + boff1 + 256);
    bf16x8 b0 = __builtin_bit_cast(bf16x8, __builtin_shufflevector(l0, h0, 0, 1, 2, 3, 4, 5, 6, 7));
    bf16x8 b1 = __builtin_bit_cast(bf16x8, __builtin_shufflevector(l1, h1, 0, 1, 2, 3, 4, 5, 6, 7));
    bf16x8 af = __builtin_bit_cast(bf16x8, a);
    acc0 = __builtin_amdgcn_mfma_f32_16x16x32_bf16(af, b0, acc0, 0, 0, 0);
    acc1 = __builtin_amdgcn_mfma_f32_16x16x32_bf16(af, b1, acc1, 0, 0, 0);
  };
  const int rg = mt * 4 + wv;
  {
    const uint16_t* xa = p.x + (size_t)(rank * 256 + rg * 16 + (lane & 15)) * 256 + (lane >> 4) * 8;
    u32x4 a[8];
#pragma unroll
    for (int k = 0; k < 8; ++k) a[k] = *(const u32x4*)(xa + k * 32);
#pragma unroll
    for (int k = 0; k < 8; ++k) step(a[k], rank * 256 + k * 32);
  }
  const unsigned target = 8u * call;
  const char* abase = myws + par + (size_t)rg * 8192 + lane * 16;
  bool dead = false;
  for (int s = 0; s < 4; ++s) {
    {
      const bool need = lane < 8 && lane != rank;
      unsigned* fl = (unsigned*)(myws + FLAG_OFF + (size_t)((lane & 7) * 4 + s) * 128);
      const unsigned long long t0 = __builtin_amdgcn_s_memrealtime();
      while (!dead) {
        unsigned v = need ? __hip_atomic_load(fl, __ATOMIC_RELAXED, __HIP_MEMORY_SCOPE_SYSTEM) : target;
        if (__all((int)(v - target) >= 0)) break;
        if (__hip_atomic_load(tmo, __ATOMIC_RELAXED, __HIP_MEMORY_SCOPE_AGENT) == call ||
            __builtin_amdgcn_s_memrealtime() - t0 > DEADLINE) {
          __hip_atomic_store(tmo, call, __ATOMIC_RELAXED, __HIP_MEMORY_SCOPE_AGENT);
          dead = true;
        }
        __builtin_amdgcn_s_sleep(1);
      }
    }
    if (dead) break;
    STAMP(3 + s);
    u32x4 a[14];
#pragma unroll
    for (int i = 0; i < 7; ++i) {
      const int src = (rank + 1 + i) & 7;
      const char* ap = abase + (size_t)src * 131072 + (s * 2) * 1024;
      asm volatile("global_load_dwordx4 %0, %1, off " LDM : "=v"(a[2 * i]) : "v"(ap) : "memory");
      asm volatile("global_load_dwordx4 %0, %1, off offset:1024 " LDM : "=v"(a[2 * i + 1]) : "v"(ap) : "memory");
    }
    asm volatile("s_waitcnt vmcnt(0)" ::: "memory");
#pragma unroll
    for (int i = 0; i < 7; ++i) {
      const int src = (rank + 1 + i) & 7;
      step(a[2 * i], src * 256 + s * 64);
      step(a[2 * i + 1], src * 256 + s * 64 + 32);
    }
  }

  STAMP(7);
  if (!dead) dead = __hip_atomic_load(tmo, __ATOMIC_RELAXED, __HIP_MEMORY_SCOPE_AGENT) == call;
  uint16_t* o = p.out + (size_t)(rg * 16 + g * 4) * 2048 + nt * 32 + (lane & 15);
#pragma unroll
  for (int r = 0; r < 4; ++r) {
    o[r * 2048] = dead ? (uint16_t)0x7FC0 : f2bf(fmaxf(acc0[r], 0.f));
    o[r * 2048 + 16] = dead ? (uint16_t)0x7FC0 : f2bf(fmaxf(acc1[r], 0.f));
  }
  if (tid == 0) __hip_atomic_store(cctr, call, __ATOMIC_RELAXED, __HIP_MEMORY_SCOPE_AGENT);
}

extern "C" void kernel_launch(int rank, int nranks, ncclComm_t comm, hipStream_t stream,
                              void* const* d_in, const int* in_sizes, int n_in,
                              void* d_out, int out_size, const int* in_dtypes, int out_dtype,
                              void* const* d_out_all, void* const* d_ws_all, size_t ws_bytes) {
  if (nranks != NR || ws_bytes < WS_NEED) return;
  Params p;
  p.x = (const uint16_t*)d_in[0]; p.w = (const uint16_t*)d_in[1]; p.out = (uint16_t*)d_out;
  for (int i = 0; i < NR; ++i) p.ws[i] = (char*)d_ws_all[i];
  p.rank = rank;
  hipLaunchKernelGGL(a2a_gemm_kernel, dim3(256), dim3(256), LDS_BYTES, stream, p);
}
```

```cpp
#include <hip/hip_runtime.h>
#include <rccl/rccl.h>
#include <cstdint>
#include <cstdio>

typedef __attribute__((ext_vector_type(8))) __bf16 bf16x8;
typedef __attribute__((ext_vector_type(4))) float f32x4;
typedef __attribute__((ext_vector_type(4))) short s16x4;
typedef __attribute__((ext_vector_type(4))) unsigned u32x4;

#define NOACQ
#define NOREL
#define LDM "sc0 sc1"
constexpr int NR = 8;
constexpr size_t STAGE_BYTES = 1u << 20;
constexpr size_t FLAG_OFF = 2 * STAGE_BYTES;
constexpr size_t CTR_OFF = FLAG_OFF + 65536;
constexpr size_t TMO_OFF = CTR_OFF + 65536;
constexpr size_t WS_NEED = TMO_OFF + 4096;
constexpr unsigned long long DEADLINE = 30000000ull;
constexpr int LDS_BYTES = 2048 * 64;

struct Params {
  const uint16_t* x; const uint16_t* w; uint16_t* out;
  char* ws[NR]; int rank;
};

__device__ __forceinline__ s16x4 trread(const char* p) {
  return __builtin_bit_cast(s16x4, __builtin_amdgcn_ds_read_tr16_b64_v4i16(
      (__attribute__((address_space(3))) s16x4*)(p)));
}

__device__ __forceinline__ uint16_t f2bf(float f) {
  unsigned u = __float_as_uint(f);
  u += 0x7FFFu + ((u >> 16) & 1u);
  return (uint16_t)(u >> 16);
}

#define STAMP(i)
__global__ __launch_bounds__(256, 1) void a2a_gemm_kernel(Params p) {
  extern __shared__ __attribute__((aligned(16))) char smem[];
  const int b = blockIdx.x, tid = threadIdx.x, lane = tid & 63, wv = tid >> 6;
  const int rank = p.rank;
  STAMP(0);
  char* myws = p.ws[rank];
  unsigned* cctr = (unsigned*)(myws + CTR_OFF + (size_t)b * 128);
  const unsigned call = __hip_atomic_load(cctr, __ATOMIC_RELAXED, __HIP_MEMORY_SCOPE_AGENT) + 1u;
  const size_t par = (size_t)(call & 1u) * STAGE_BYTES;
  unsigned* tmo = (unsigned*)(myws + TMO_OFF);

  const int xcd = b & 7, idx = b >> 3;
  const int nt = xcd * 8 + (idx & 7), mt = idx >> 3;
  auto WLOAD = [&]() {
    const int row = lane >> 2, piece = (lane & 3) ^ (((row >> 3) & 1) << 1);
    const uint16_t* wsrc = p.w + (size_t)row * 2048 + nt * 32 + piece * 8;
#pragma unroll 8
    for (int i = 0; i < 32; ++i) {
      const int blk = i * 4 + wv;
      __builtin_amdgcn_global_load_lds((const unsigned*)(wsrc + (size_t)blk * 16 * 2048),
          (__attribute__((address_space(3))) unsigned*)(smem + blk * 1024), 16, 0, 0);
    }
  };
  if (b < 224) {
    const int s = b / 56, rem = b % 56, pi = rem % 7, part = rem / 7;
    const unsigned peer = (unsigned)(rank + 1 + pi) & 7u;
    const int rg = part * 2 + (wv >> 1), ks = s * 2 + (wv & 1);
    const u32x4 v = *(const u32x4*)(p.x + (size_t)(peer * 256 + rg * 16 + (lane & 15)) * 256 + ks * 32 + (lane >> 4) * 8);
    char* pw = p.ws[peer];
    asm volatile("global_store_dwordx4 %0, %1, off sc0 sc1" :: "v"(pw + par + (size_t)((rank * 16 + rg) * 8 + ks) * 1024 + lane * 16), "v"(v) : "memory");
    WLOAD();
    asm volatile("s_waitcnt vmcnt(32)" ::: "memory");
    __builtin_amdgcn_s_barrier();
    if (tid == 0) {
      __hip_atomic_fetch_add((unsigned*)(pw + FLAG_OFF + (size_t)(rank * 4 + s) * 128), 1u,
                             __ATOMIC_RELAXED, __HIP_MEMORY_SCOPE_SYSTEM);
    }
  } else WLOAD();

  STAMP(1);
  asm volatile("s_waitcnt vmcnt(0)" ::: "memory");
  __syncthreads();
  STAMP(2);
  f32x4 acc0 = {0, 0, 0, 0}, acc1 = {0, 0, 0, 0};
  const int g = lane >> 4;
  const int boff0 = (8 * g + ((lane >> 2) & 3)) * 64 + ((g & 1) * 32) + (lane & 3) * 8;
  const int boff1 = boff0 ^ 32;
  auto step = [&](const u32x4& a, int kk) {
    const char* bp = smem + kk * 64;
    s16x4 l0 = trread(bp + boff0), h0 = trread(bp + boff0 + 256);
    s16x4 l1 = trread(bp + boff1), h1 = trread(bp + boff1 + 256);
    bf16x8 b0 = __builtin_bit_cast(bf16x8, __builtin_shufflevector(l0, h0, 0, 1, 2, 3, 4, 5, 6, 7));
    bf16x8 b1 = __builtin_bit_cast(bf16x8, __builtin_shufflevector(l1, h1, 0, 1, 2, 3, 4, 5, 6, 7));
    bf16x8 af = __builtin_bit_cast(bf16x8, a);
    acc0 = __builtin_amdgcn_mfma_f32_16x16x32_bf16(af, b0, acc0, 0, 0, 0);
    acc1 = __builtin_amdgcn_mfma_f32_16x16x32_bf16(af, b1, acc1, 0, 0, 0);
  };
  const int rg = mt * 4 + wv;
  {
    const uint16_t* xa = p.x + (size_t)(rank * 256 + rg * 16 + (lane & 15)) * 256 + (lane >> 4) * 8;
    u32x4 a[8];
#pragma unroll
    for (int k = 0; k < 8; ++k) a[k] = *(const u32x4*)(xa + k * 32);
#pragma unroll
    for (int k = 0; k < 8; ++k) step(a[k], rank * 256 + k * 32);
  }
  const unsigned target = 8u * call;
  const char* abase = myws + par + (size_t)rg * 8192 + lane * 16;
  bool dead = false;
  auto WAIT = [&](int s) {
      const bool need = lane < 8 && lane != rank;
      unsigned* fl = (unsigned*)(myws + FLAG_OFF + (size_t)((lane & 7) * 4 + s) * 128);
      const unsigned long long t0 = __builtin_amdgcn_s_memrealtime();
      while (!dead) {
        unsigned v = need ? __hip_atomic_load(fl, __ATOMIC_RELAXED, __HIP_MEMORY_SCOPE_SYSTEM) : target;
        if (__all((int)(v - target) >= 0)) break;
        if (__hip_atomic_load(tmo, __ATOMIC_RELAXED, __HIP_MEMORY_SCOPE_AGENT) == call ||
            __builtin_amdgcn_s_memrealtime() - t0 > DEADLINE) {
          __hip_atomic_store(tmo, call, __ATOMIC_RELAXED, __HIP_MEMORY_SCOPE_AGENT);
          dead = true;
        }
        __builtin_amdgcn_s_sleep(1);
      }
  };
  auto ISSUE = [&](u32x4* a, int s) {
#pragma unroll
    for (int i = 0; i < 7; ++i) {
      const int src = (rank + 1 + i) & 7;
      const char* ap = abase + (size_t)src * 131072 + (s * 2) * 1024;
      asm volatile("global_load_dwordx4 %0, %1, off " LDM : "=v"(a[2 * i]) : "v"(ap) : "memory");
      asm volatile("global_load_dwordx4 %0, %1, off offset:1024 " LDM : "=v"(a[2 * i + 1]) : "v"(ap) : "memory");
    }
  };
  u32x4 abuf[2][14];
  WAIT(0);
  ISSUE(abuf[0], 0);
#pragma unroll
  for (int s = 0; s < 4; ++s) {
    if (s < 3) { WAIT(s + 1); ISSUE(abuf[(s + 1) & 1], s + 1); asm volatile("s_waitcnt vmcnt(14)" ::: "memory"); }
    else asm volatile("s_waitcnt vmcnt(0)" ::: "memory");
    STAMP(3 + s);
#pragma unroll
    for (int i = 0; i < 7; ++i) {
      const int src = (rank + 1 + i) & 7;
      step(abuf[s & 1][2 * i], src * 256 + s * 64);
      step(abuf[s & 1][2 * i + 1], src * 256 + s * 64 + 32);
    }
  }

  STAMP(7);
  if (!dead) dead = __hip_atomic_load(tmo, __ATOMIC_RELAXED, __HIP_MEMORY_SCOPE_AGENT) == call;
  uint16_t* o = p.out + (size_t)(rg * 16 + g * 4) * 2048 + nt * 32 + (lane & 15);
#pragma unroll
  for (int r = 0; r < 4; ++r) {
    o[r * 2048] = dead ? (uint16_t)0x7FC0 : f2bf(fmaxf(acc0[r], 0.f));
    o[r * 2048 + 16] = dead ? (uint16_t)0x7FC0 : f2bf(fmaxf(acc1[r], 0.f));
  }
  if (tid == 0) __hip_atomic_store(cctr, call, __ATOMIC_RELAXED, __HIP_MEMORY_SCOPE_AGENT);
}

extern "C" void kernel_launch(int rank, int nranks, ncclComm_t comm, hipStream_t stream,
                              void* const* d_in, const int* in_sizes, int n_in,
                              void* d_out, int out_size, const int* in_dtypes, int out_dtype,
                              void* const* d_out_all, void* const* d_ws_all, size_t ws_bytes) {
  if (nranks != NR || ws_bytes < WS_NEED) return;
  Params p;
  p.x = (const uint16_t*)d_in[0]; p.w = (const uint16_t*)d_in[1]; p.out = (uint16_t*)d_out;
  for (int i = 0; i < NR; ++i) p.ws[i] = (char*)d_ws_all[i];
  p.rank = rank;
  hipLaunchKernelGGL(a2a_gemm_kernel, dim3(256), dim3(256), LDS_BYTES, stream, p);
}
```
